```python
import numpy as np

kernel_name = "gpu_a2a_gemm_m4096_k4096_n16384_bf16_qe4m3"


ABSMAX_RTOL = 0.09

COMPUTE_DTYPE = "bfloat16"
EPILOGUE = "quant_e4m3"
IN_QUANT = None
OUT_QUANT = "e4m3"
IN_DTYPES = {"x": "bfloat16", "w_mat": "bfloat16"}
WORLD_SIZE = 8

def _to_bf16(x):
    u = np.ascontiguousarray(x, dtype=np.float32).view(np.uint32)
    return ((u + 0x7FFF + ((u >> 16) & 1)) & 0xFFFF0000).view(np.float32)

def setup_inputs():

    x = _to_bf16(np.random.randn(4096, 4096).astype(np.float32))
    w_mat = _to_bf16((np.random.randn(4096, 16384) / np.sqrt(4096)).astype(np.float32))
    return {"x": x, "w_mat": w_mat}

def _snap_fp8(x, mant_bits, max_abs, min_normal):
    a = np.abs(x.astype(np.float64))
    _m, _e = np.frexp(a)
    q = np.where(a >= min_normal, np.ldexp(1.0, _e - 1 - mant_bits),
                 min_normal * 2.0 ** -mant_bits)
    snapped = np.minimum(np.round(a / q) * q, max_abs)
    return (np.sign(x) * snapped).astype(np.float32)
def _quant_dequant(y, amax):
    scale = amax / 448.0
    q = _snap_fp8(y / scale, 3, 448.0, 2.0 ** -6)
    return (q * scale).astype(np.float32)
def _epilogue(y):
    return _quant_dequant(y, np.abs(y).max())

def reference(x, w_mat):
    y = (x @ w_mat).astype(np.float32)
    return _epilogue(y)
```

```cpp
#include <hip/hip_runtime.h>
#include <rccl/rccl.h>
#include <cstdio>
#include <cstdint>

typedef __attribute__((ext_vector_type(8))) short bf16x8;
typedef __attribute__((ext_vector_type(4))) short s16x4;
typedef __attribute__((ext_vector_type(4))) float f32x4;
typedef __attribute__((ext_vector_type(4))) unsigned u32x4;
typedef unsigned long long u64;

constexpr int NR = 8;
constexpr int KSL = 512;
constexpr int NTOT = 16384;
constexpr int BK = 64;
constexpr int NWG = 256, NT = 512;
constexpr int NKT = 64;
constexpr int STG = 48 * 1024;
constexpr int CTL = 3 * STG;
constexpr int LDS_BYTES = CTL + 256;
constexpr u64 DEADLINE = 50000000ull;

constexpr size_t W_CALL = 0, W_TMO = 128, W_XCNT = 256  , W_LMAX = 2048,
                 W_FIN = 2176, W_SLOT = 2304  , W_X = 1 << 20;

struct Ptrs { char* ws[NR]; };

#define RLX __ATOMIC_RELAXED
#define SYS __HIP_MEMORY_SCOPE_SYSTEM
#define AGT __HIP_MEMORY_SCOPE_AGENT

__device__ __forceinline__ bool give_up(unsigned* tmo, unsigned call, u64 t0) {
  if (__hip_atomic_load(tmo, RLX, AGT) == call || __builtin_amdgcn_s_memrealtime() - t0 > DEADLINE) {
    __hip_atomic_store(tmo, call, RLX, AGT);
    return true;
  }
  return false;
}

__device__ __forceinline__ void wait_ge(const unsigned* flag, unsigned target, unsigned* tmo, unsigned call) {
  u64 t0 = __builtin_amdgcn_s_memrealtime();
  while (__hip_atomic_load(flag, RLX, SYS) < target) {
    if (give_up(tmo, call, t0)) break;
    __builtin_amdgcn_s_sleep(2);
  }
  __builtin_amdgcn_fence(__ATOMIC_ACQUIRE, "");
}

__global__ __launch_bounds__(NT, 2)
void a2a_gemm_q(int rank, int nranks, const unsigned short* __restrict__ x,
                const unsigned short* __restrict__ w, unsigned char* __restrict__ out, Ptrs P) {
  extern __shared__ __attribute__((aligned(16))) char smem[];
  const int tid = threadIdx.x, lane = tid & 63, wid = tid >> 6;
  const int b = blockIdx.x;
  char* ws = P.ws[rank];
  unsigned* tmo = (unsigned*)(ws + W_TMO);
  volatile unsigned* ctl = (volatile unsigned*)(smem + CTL);

  if (tid == 0) {
    unsigned t = __hip_atomic_fetch_add((unsigned*)(ws + W_CALL), 1u, RLX, AGT);
    ctl[0] = t / NWG + 1;
  }
  if (nranks > 1) {
#pragma unroll
    for (int i = 0; i < 2; ++i) {
      int idx = tid + i * NT;
      unsigned p = (unsigned)(idx >> 7);
      if (p < (unsigned)nranks && (int)p != rank) {
        const u32x4 v = *(const u32x4*)(x + (size_t)p * KSL * KSL + b * 1024 + (idx & 127) * 8);
        *(u32x4*)(P.ws[p] + W_X + (size_t)rank * KSL * KSL * 2 + b * 2048 + (idx & 127) * 16) = v;
      }
    }
    asm volatile("s_waitcnt vmcnt(0)" ::: "memory");
  }
  __syncthreads();
  const unsigned call = ctl[0];
  if (nranks > 1 && tid < nranks && tid != rank) {
    __builtin_amdgcn_fence(__ATOMIC_RELEASE, "");
    asm volatile("s_waitcnt vmcnt(0)" ::: "memory");
    __hip_atomic_fetch_add((unsigned*)(P.ws[tid] + W_XCNT + 128 * rank), 1u, RLX, SYS);
  }

  const int xcd = b & 7, j = b >> 3;
  const int mt = j & 1, nt = xcd * 16 + (j >> 1);
  const int m0 = mt * 256, n0 = nt * 128;
  const int wm = wid >> 1, wn = wid & 1;

  int aoff[4], boff[2];
#pragma unroll
  for (int i = 0; i < 4; ++i) {
    int by = i * 8192 + tid * 16, row = by >> 7, c = ((by >> 4) & 7) ^ (row & 7);
    aoff[i] = (m0 + row) * KSL + c * 8;
  }
#pragma unroll
  for (int i = 0; i < 2; ++i) {
    int by = i * 8192 + tid * 16, k = by >> 8, cp = (by >> 4) & 15;
    int s = (k & 3) | (((k >> 3) & 1) << 2);
    int c = (((cp >> 1) ^ s) << 1) | (cp & 1);
    boff[i] = k * NTOT + n0 + c * 8;
  }
  const int fr = lane & 15, fg = lane >> 4;
  int ardo[2];
#pragma unroll
  for (int kk = 0; kk < 2; ++kk) {
    int row = wm * 64 + fr;
    ardo[kk] = row * 128 + (((kk * 4 + fg) ^ (row & 7)) << 4);
  }
  const int q = (lane >> 2) & 3, p4 = lane & 3;
  int brdo[4];
  {
    int s = q | ((fg & 1) << 2);
#pragma unroll
    for (int ni = 0; ni < 4; ++ni) {
      int c = wn * 8 + ni * 2 + (p4 >> 1);
      brdo[ni] = 32768 + (8 * fg + q) * 256 + ((((c >> 1) ^ s) << 1 | (c & 1)) << 4) + 8 * (p4 & 1);
    }
  }

  f32x4 acc[4][4];
#pragma unroll
  for (int a = 0; a < 4; ++a)
#pragma unroll
    for (int c = 0; c < 4; ++c) acc[a][c] = (f32x4){0.f, 0.f, 0.f, 0.f};

  auto stage = [&](int t, int buf) {
    int sl = (rank + (t >> 3)) & 7;
    const unsigned short* abase = (sl == rank) ? (x + (size_t)rank * KSL * KSL)
                                               : (const unsigned short*)(ws + W_X) + (size_t)sl * KSL * KSL;
    abase += (t & 7) * BK;
    const unsigned short* bbase = w + (size_t)(sl * KSL + (t & 7) * BK) * NTOT;
    char* lb = smem + buf * STG;
#pragma unroll
    for (int i = 0; i < 4; ++i)
      __builtin_amdgcn_global_load_lds((const unsigned*)(abase + aoff[i]), (unsigned*)(lb + i * 8192 + tid * 16), 16, 0, 0);
#pragma unroll
    for (int i = 0; i < 2; ++i)
      __builtin_amdgcn_global_load_lds((const unsigned*)(bbase + boff[i]), (unsigned*)(lb + 32768 + i * 8192 + tid * 16), 16, 0, 0);
  };

  stage(0, 0);
  stage(1, 1);
  int buf = 0;
  for (int t = 0; t < NKT; ++t) {
    if (t < NKT - 1) asm volatile("s_waitcnt vmcnt(6)" ::: "memory");
    else asm volatile("s_waitcnt vmcnt(0)" ::: "memory");
    __builtin_amdgcn_s_barrier();
    if (t + 2 < NKT) {
      if (((t + 2) & 7) == 0 && nranks > 1) {
        int sl = (rank + ((t + 2) >> 3)) & 7;
        wait_ge((const unsigned*)(ws + W_XCNT + 128 * sl), call * NWG, tmo, call);
      }
      int nb = buf + 2; if (nb >= 3) nb -= 3;
      stage(t + 2, nb);
    }
    const char* lb = smem + buf * STG;
#pragma unroll
    for (int kk = 0; kk < 2; ++kk) {
      bf16x8 xa[4], wb[4];
#pragma unroll
      for (int mi = 0; mi < 4; ++mi) xa[mi] = *(const bf16x8*)(lb + ardo[kk] + mi * 2048);
#pragma unroll
      for (int ni = 0; ni < 4; ++ni) {
        typedef __attribute__((ext_vector_type(4))) short v4i16_t;
        v4i16_t lo = __builtin_amdgcn_ds_read_tr16_b64_v4i16((__attribute__((address_space(3))) v4i16_t*)(lb + brdo[ni] + kk * 8192));
        v4i16_t hi = __builtin_amdgcn_ds_read_tr16_b64_v4i16((__attribute__((address_space(3))) v4i16_t*)(lb + brdo[ni] + kk * 8192 + 1024));
        wb[ni] = (bf16x8){lo[0], lo[1], lo[2], lo[3], hi[0], hi[1], hi[2], hi[3]};
      }
      __builtin_amdgcn_s_setprio(1);
#pragma unroll
      for (int ni = 0; ni < 4; ++ni)
#pragma unroll
        for (int mi = 0; mi < 4; ++mi)
          acc[ni][mi] = __builtin_amdgcn_mfma_f32_16x16x32_bf16(wb[ni], xa[mi], acc[ni][mi], 0, 0, 0);
      __builtin_amdgcn_s_setprio(0);
    }
    buf = buf + 1; if (buf >= 3) buf = 0;
  }

  float mx = 0.f;
#pragma unroll
  for (int a = 0; a < 4; ++a)
#pragma unroll
    for (int c = 0; c < 4; ++c)
#pragma unroll
      for (int e = 0; e < 4; ++e) mx = fmaxf(mx, fabsf(acc[a][c][e]));
#pragma unroll
  for (int o = 32; o >= 1; o >>= 1) mx = fmaxf(mx, __shfl_xor(mx, o));
  if (lane == 0) ctl[8 + wid] = __float_as_uint(mx);
  __syncthreads();
  if (wid == 0) {
    if (lane == 0) {
      unsigned m = 0;
      for (int i = 0; i < 8; ++i) m = max(m, ctl[8 + i]);
      __hip_atomic_fetch_max((u64*)(ws + W_LMAX), ((u64)call << 32) | m, RLX, AGT);
      unsigned tk = __hip_atomic_fetch_add((unsigned*)(ws + W_FIN), 1u, __ATOMIC_ACQ_REL, AGT);
      if (tk == call * NWG - 1) {
        u64 v = __hip_atomic_load((u64*)(ws + W_LMAX), RLX, AGT);
        for (int p = 0; p < nranks; ++p) __hip_atomic_store((u64*)(P.ws[p] + W_SLOT + 128 * rank), v, RLX, SYS);
      }
    }
    unsigned m = 0;
    if (lane < nranks) {
      u64 t0 = __builtin_amdgcn_s_memrealtime();
      u64 v;
      while (((v = __hip_atomic_load((u64*)(ws + W_SLOT + 128 * lane), RLX, SYS)) >> 32) < call) {
        if (give_up(tmo, call, t0)) break;
        __builtin_amdgcn_s_sleep(1);
      }
      m = (unsigned)v;
    }
#pragma unroll
    for (int o = 4; o >= 1; o >>= 1) m = max(m, (unsigned)__shfl_xor((int)m, o));
    if (lane == 0) { ctl[1] = m; ctl[2] = (__hip_atomic_load(tmo, RLX, AGT) == call); }
  }
  __syncthreads();
  const float amax = __uint_as_float(ctl[1]);
  const bool bad = ctl[2] != 0;
  float sc = amax / 448.0f;
  if (!(sc > 0.f)) sc = 1.f;

#pragma unroll
  for (int ni = 0; ni < 4; ++ni)
#pragma unroll
    for (int mi = 0; mi < 4; ++mi) {
      float v0 = fminf(fmaxf(acc[ni][mi][0] / sc, -448.f), 448.f);
      float v1 = fminf(fmaxf(acc[ni][mi][1] / sc, -448.f), 448.f);
      float v2 = fminf(fmaxf(acc[ni][mi][2] / sc, -448.f), 448.f);
      float v3 = fminf(fmaxf(acc[ni][mi][3] / sc, -448.f), 448.f);
      int pk = __builtin_amdgcn_cvt_pk_fp8_f32(v0, v1, 0, false);
      pk = __builtin_amdgcn_cvt_pk_fp8_f32(v2, v3, pk, true);
      if (bad) pk = 0x7f7f7f7f;
      *(int*)(smem + (wm * 64 + mi * 16 + fr) * 144 + wn * 64 + ni * 16 + fg * 4) = pk;
    }
  __syncthreads();
#pragma unroll
  for (int i = 0; i < 4; ++i) {
    int row = i * 64 + (tid >> 3), ch = tid & 7;
    u32x4 v = *(const u32x4*)(smem + row * 144 + ch * 16);
    *(u32x4*)(out + (size_t)(m0 + row) * NTOT + n0 + ch * 16) = v;
  }
}

extern "C" void kernel_launch(int rank, int nranks, ncclComm_t comm, hipStream_t stream,
                              void* const* d_in, const int* in_sizes, int n_in,
                              void* d_out, int out_size, const int* in_dtypes, int out_dtype,
                              void* const* d_out_all, void* const* d_ws_all, size_t ws_bytes) {
  static bool init = false;
  if (!init) {
    (void)hipFuncSetAttribute((const void*)a2a_gemm_q, hipFuncAttributeMaxDynamicSharedMemorySize, LDS_BYTES);
    init = true;
  }
  Ptrs P;
  for (int p = 0; p < NR; ++p) P.ws[p] = (char*)d_ws_all[p < nranks ? p : rank];
  a2a_gemm_q<<<NWG, NT, LDS_BYTES, stream>>>(rank, nranks, (const unsigned short*)d_in[0],
                                             (const unsigned short*)d_in[1], (unsigned char*)d_out, P);
}
```

```cpp
#include <hip/hip_runtime.h>
#include <rccl/rccl.h>
#include <cstdio>
#include <cstdint>

typedef __attribute__((ext_vector_type(8))) short bf16x8;
typedef __attribute__((ext_vector_type(4))) short s16x4;
typedef __attribute__((ext_vector_type(4))) float f32x4;
typedef __attribute__((ext_vector_type(4))) unsigned u32x4;
typedef unsigned long long u64;

constexpr int NR = 8;
constexpr int KSL = 512;
constexpr int NTOT = 16384;
constexpr int BK = 64;
constexpr int NWG = 256, NT = 512;
constexpr int NKT = 64;
constexpr int STG = 48 * 1024;
constexpr int CTL = 3 * STG;
constexpr int LDS_BYTES = CTL + 256;
constexpr u64 DEADLINE = 50000000ull;

constexpr size_t W_CALL = 0, W_TMO = 128, W_XCNT = 256  , W_LMAX = 2048,
                 W_FIN = 2176, W_SLOT = 2304  , W_X = 1 << 20;

struct Ptrs { char* ws[NR]; };

#define RLX __ATOMIC_RELAXED
#define SYS __HIP_MEMORY_SCOPE_SYSTEM
#define AGT __HIP_MEMORY_SCOPE_AGENT

__device__ __forceinline__ bool give_up(unsigned* tmo, unsigned call, u64 t0) {
  if (__hip_atomic_load(tmo, RLX, AGT) == call || __builtin_amdgcn_s_memrealtime() - t0 > DEADLINE) {
    __hip_atomic_store(tmo, call, RLX, AGT);
    return true;
  }
  return false;
}

__device__ __forceinline__ void wait_ge(const unsigned* flag, unsigned target, unsigned* tmo, unsigned call) {
  u64 t0 = __builtin_amdgcn_s_memrealtime();
  while (__hip_atomic_load(flag, RLX, SYS) < target) {
    if (give_up(tmo, call, t0)) break;
    __builtin_amdgcn_s_sleep(2);
  }
  __builtin_amdgcn_fence(__ATOMIC_ACQUIRE, "");
}

__global__ __launch_bounds__(NT, 2)
void a2a_gemm_q(int rank, int nranks, const unsigned short* __restrict__ x,
                const unsigned short* __restrict__ w, unsigned char* __restrict__ out, Ptrs P) {
  extern __shared__ __attribute__((aligned(16))) char smem[];
  const int tid = threadIdx.x, lane = tid & 63, wid = tid >> 6;
  const int b = blockIdx.x;
  char* ws = P.ws[rank];
  unsigned* tmo = (unsigned*)(ws + W_TMO);
  volatile unsigned* ctl = (volatile unsigned*)(smem + CTL);

  if (tid == 0) {
    unsigned t = __hip_atomic_fetch_add((unsigned*)(ws + W_CALL), 1u, RLX, AGT);
    ctl[0] = t / NWG + 1;
  }
  if (nranks > 1) {
#pragma unroll
    for (int i = 0; i < 2; ++i) {
      int idx = tid + i * NT;
      unsigned p = (unsigned)(idx >> 7);
      if (p < (unsigned)nranks && (int)p != rank) {
        const u32x4 v = *(const u32x4*)(x + (size_t)p * KSL * KSL + b * 1024 + (idx & 127) * 8);
        *(u32x4*)(P.ws[p] + W_X + (size_t)rank * KSL * KSL * 2 + b * 2048 + (idx & 127) * 16) = v;
      }
    }
    asm volatile("s_waitcnt vmcnt(0)" ::: "memory");
  }
  __syncthreads();
  const unsigned call = ctl[0];
  if (nranks > 1 && tid < nranks && tid != rank) {
    __builtin_amdgcn_fence(__ATOMIC_RELEASE, "");
    asm volatile("s_waitcnt vmcnt(0)" ::: "memory");
    __hip_atomic_fetch_add((unsigned*)(P.ws[tid] + W_XCNT + 128 * rank), 1u, RLX, SYS);
  }

  const int xcd = b & 7, j = b >> 3;
  const int mt = j & 1, nt = xcd * 16 + (j >> 1);
  const int m0 = mt * 256, n0 = nt * 128;
  const int wm = wid >> 1, wn = wid & 1;

  int aoff[4], boff[2];
#pragma unroll
  for (int i = 0; i < 4; ++i) {
    int by = i * 8192 + tid * 16, row = by >> 7, c = ((by >> 4) & 7) ^ (row & 7);
    aoff[i] = (m0 + row) * KSL + c * 8;
  }
#pragma unroll
  for (int i = 0; i < 2; ++i) {
    int by = i * 8192 + tid * 16, k = by >> 8, cp = (by >> 4) & 15;
    int s = (k & 3) | (((k >> 3) & 1) << 2);
    int c = (((cp >> 1) ^ s) << 1) | (cp & 1);
    boff[i] = k * NTOT + n0 + c * 8;
  }
  const int fr = lane & 15, fg = lane >> 4;
  int ardo[2];
#pragma unroll
  for (int kk = 0; kk < 2; ++kk) {
    int row = wm * 64 + fr;
    ardo[kk] = row * 128 + (((kk * 4 + fg) ^ (row & 7)) << 4);
  }
  const int q = (lane >> 2) & 3, p4 = lane & 3;
  int brdo[4];
  {
    int s = q | ((fg & 1) << 2);
#pragma unroll
    for (int ni = 0; ni < 4; ++ni) {
      int c = wn * 8 + ni * 2 + (p4 >> 1);
      brdo[ni] = 32768 + (8 * fg + q) * 256 + ((((c >> 1) ^ s) << 1 | (c & 1)) << 4) + 8 * (p4 & 1);
    }
  }

  f32x4 acc[4][4];
#pragma unroll
  for (int a = 0; a < 4; ++a)
#pragma unroll
    for (int c = 0; c < 4; ++c) acc[a][c] = (f32x4){0.f, 0.f, 0.f, 0.f};

  auto stage = [&](int t, int buf) {
    int sl = (rank + (t >> 3)) & 7;
    const unsigned short* abase = (sl == rank) ? (x + (size_t)rank * KSL * KSL)
                                               : (const unsigned short*)(ws + W_X) + (size_t)sl * KSL * KSL;
    abase += (t & 7) * BK;
    const unsigned short* bbase = w + (size_t)(sl * KSL + (t & 7) * BK) * NTOT;
    char* lb = smem + buf * STG;
#pragma unroll
    for (int i = 0; i < 4; ++i)
      __builtin_amdgcn_global_load_lds((const unsigned*)(abase + aoff[i]), (unsigned*)(lb + i * 8192 + tid * 16), 16, 0, 0);
#pragma unroll
    for (int i = 0; i < 2; ++i)
      __builtin_amdgcn_global_load_lds((const unsigned*)(bbase + boff[i]), (unsigned*)(lb + 32768 + i * 8192 + tid * 16), 16, 0, 0);
  };

  bf16x8 xa[2][2][4];
  s16x4 wl[2][2][4], wh[2][2][4];
  auto rdfrag = [&](int set, int bufi) {
    const unsigned lb = (unsigned)(size_t)smem + bufi * STG;
#pragma unroll
    for (int kk = 0; kk < 2; ++kk) {
      unsigned aa = lb + ardo[kk];
      asm volatile("ds_read_b128 %0, %4\n\tds_read_b128 %1, %4 offset:2048\n\tds_read_b128 %2, %4 offset:4096\n\tds_read_b128 %3, %4 offset:6144"
                   : "=&v"(xa[set][kk][0]), "=&v"(xa[set][kk][1]), "=&v"(xa[set][kk][2]), "=&v"(xa[set][kk][3]) : "v"(aa));
#pragma unroll
      for (int ni = 0; ni < 4; ++ni) {
        unsigned ba = lb + brdo[ni];
        if (kk == 0)
          asm volatile("ds_read_b64_tr_b16 %0, %2\n\tds_read_b64_tr_b16 %1, %2 offset:1024" : "=&v"(wl[set][kk][ni]), "=&v"(wh[set][kk][ni]) : "v"(ba));
        else
          asm volatile("ds_read_b64_tr_b16 %0, %2 offset:8192\n\tds_read_b64_tr_b16 %1, %2 offset:9216" : "=&v"(wl[set][kk][ni]), "=&v"(wh[set][kk][ni]) : "v"(ba));
      }
    }
  };
  auto mma = [&](int set) {
    __builtin_amdgcn_s_setprio(1);
#pragma unroll
    for (int kk = 0; kk < 2; ++kk)
#pragma unroll
      for (int ni = 0; ni < 4; ++ni) {
        bf16x8 wb = (bf16x8){wl[set][kk][ni][0], wl[set][kk][ni][1], wl[set][kk][ni][2], wl[set][kk][ni][3],
                             wh[set][kk][ni][0], wh[set][kk][ni][1], wh[set][kk][ni][2], wh[set][kk][ni][3]};
#pragma unroll
        for (int mi = 0; mi < 4; ++mi)
          acc[ni][mi] = __builtin_amdgcn_mfma_f32_16x16x32_bf16(wb, xa[set][kk][mi], acc[ni][mi], 0, 0, 0);
      }
    __builtin_amdgcn_s_setprio(0);
  };
  auto iter = [&](int t, int set) {
    if (t + 2 < NKT) asm volatile("s_waitcnt vmcnt(6) lgkmcnt(0)" ::: "memory");
    else asm volatile("s_waitcnt vmcnt(0) lgkmcnt(0)" ::: "memory");
    __builtin_amdgcn_sched_barrier(0);
    __builtin_amdgcn_s_barrier();
    if (t + 3 < NKT) {
      if (((t + 3) & 7) == 0 && nranks > 1) {
        int sl = (rank + ((t + 3) >> 3)) & 7;
        wait_ge((const unsigned*)(ws + W_XCNT + 128 * sl), call * NWG, tmo, call);
      }
      stage(t + 3, t % 3);
    }
    if (t + 1 < NKT) rdfrag(set ^ 1, (t + 1) % 3);
    __builtin_amdgcn_sched_barrier(0);
    mma(set);
  };
  stage(0, 0);
  stage(1, 1);
  stage(2, 2);
  asm volatile("s_waitcnt vmcnt(12)" ::: "memory");
  __builtin_amdgcn_s_barrier();
  rdfrag(0, 0);
  for (int t = 0; t < NKT; t += 2) {
    iter(t, 0);
    iter(t + 1, 1);
  }

  float mx = 0.f;
#pragma unroll
  for (int a = 0; a < 4; ++a)
#pragma unroll
    for (int c = 0; c < 4; ++c)
#pragma unroll
      for (int e = 0; e < 4; ++e) mx = fmaxf(mx, fabsf(acc[a][c][e]));
#pragma unroll
  for (int o = 32; o >= 1; o >>= 1) mx = fmaxf(mx, __shfl_xor(mx, o));
  if (lane == 0) ctl[8 + wid] = __float_as_uint(mx);
  __syncthreads();
  if (wid == 0) {
    if (lane == 0) {
      unsigned m = 0;
      for (int i = 0; i < 8; ++i) m = max(m, ctl[8 + i]);
      __hip_atomic_fetch_max((u64*)(ws + W_LMAX), ((u64)call << 32) | m, RLX, AGT);
      unsigned tk = __hip_atomic_fetch_add((unsigned*)(ws + W_FIN), 1u, __ATOMIC_ACQ_REL, AGT);
      if (tk == call * NWG - 1) {
        u64 v = __hip_atomic_load((u64*)(ws + W_LMAX), RLX, AGT);
        for (int p = 0; p < nranks; ++p) __hip_atomic_store((u64*)(P.ws[p] + W_SLOT + 128 * rank), v, RLX, SYS);
      }
    }
    unsigned m = 0;
    if (lane < nranks) {
      u64 t0 = __builtin_amdgcn_s_memrealtime();
      u64 v;
      while (((v = __hip_atomic_load((u64*)(ws + W_SLOT + 128 * lane), RLX, SYS)) >> 32) < call) {
        if (give_up(tmo, call, t0)) break;
        __builtin_amdgcn_s_sleep(1);
      }
      m = (unsigned)v;
    }
#pragma unroll
    for (int o = 4; o >= 1; o >>= 1) m = max(m, (unsigned)__shfl_xor((int)m, o));
    if (lane == 0) { ctl[1] = m; ctl[2] = (__hip_atomic_load(tmo, RLX, AGT) == call); }
  }
  __syncthreads();
  const float amax = __uint_as_float(ctl[1]);
  const bool bad = ctl[2] != 0;
  float sc = amax / 448.0f;
  if (!(sc > 0.f)) sc = 1.f;

#pragma unroll
  for (int ni = 0; ni < 4; ++ni)
#pragma unroll
    for (int mi = 0; mi < 4; ++mi) {
      float v0 = fminf(fmaxf(acc[ni][mi][0] / sc, -448.f), 448.f);
      float v1 = fminf(fmaxf(acc[ni][mi][1] / sc, -448.f), 448.f);
      float v2 = fminf(fmaxf(acc[ni][mi][2] / sc, -448.f), 448.f);
      float v3 = fminf(fmaxf(acc[ni][mi][3] / sc, -448.f), 448.f);
      int pk = __builtin_amdgcn_cvt_pk_fp8_f32(v0, v1, 0, false);
      pk = __builtin_amdgcn_cvt_pk_fp8_f32(v2, v3, pk, true);
      if (bad) pk = 0x7f7f7f7f;
      *(int*)(smem + (wm * 64 + mi * 16 + fr) * 144 + wn * 64 + ni * 16 + fg * 4) = pk;
    }
  __syncthreads();
#pragma unroll
  for (int i = 0; i < 4; ++i) {
    int row = i * 64 + (tid >> 3), ch = tid & 7;
    u32x4 v = *(const u32x4*)(smem + row * 144 + ch * 16);
    *(u32x4*)(out + (size_t)(m0 + row) * NTOT + n0 + ch * 16) = v;
  }
}

extern "C" void kernel_launch(int rank, int nranks, ncclComm_t comm, hipStream_t stream,
                              void* const* d_in, const int* in_sizes, int n_in,
                              void* d_out, int out_size, const int* in_dtypes, int out_dtype,
                              void* const* d_out_all, void* const* d_ws_all, size_t ws_bytes) {
  static bool init = false;
  if (!init) {
    (void)hipFuncSetAttribute((const void*)a2a_gemm_q, hipFuncAttributeMaxDynamicSharedMemorySize, LDS_BYTES);
    init = true;
  }
  Ptrs P;
  for (int p = 0; p < NR; ++p) P.ws[p] = (char*)d_ws_all[p < nranks ? p : rank];
  a2a_gemm_q<<<NWG, NT, LDS_BYTES, stream>>>(rank, nranks, (const unsigned short*)d_in[0],
                                             (const unsigned short*)d_in[1], (unsigned char*)d_out, P);
}
```

```cpp
#include <hip/hip_runtime.h>
#include <rccl/rccl.h>
#include <cstdio>
#include <cstdint>

typedef __attribute__((ext_vector_type(8))) short bf16x8;
typedef __attribute__((ext_vector_type(4))) short s16x4;
typedef __attribute__((ext_vector_type(4))) float f32x4;
typedef __attribute__((ext_vector_type(4))) unsigned u32x4;
typedef unsigned long long u64;

constexpr int NR = 8;
constexpr int KSL = 512;
constexpr int NTOT = 16384;
constexpr int BK = 64;
constexpr int NWG = 256, NT = 512;
constexpr int NKT = 64;
constexpr int STG = 48 * 1024;
constexpr int CTL = 3 * STG;
constexpr int LDS_BYTES = CTL + 256;
constexpr u64 DEADLINE = 50000000ull;

constexpr size_t W_CALL = 0, W_TMO = 128, W_XCNT = 256  , W_LMAX = 2048,
                 W_FIN = 2176, W_SLOT = 2304  , W_X = 1 << 20;

struct Ptrs { char* ws[NR]; };

#define RLX __ATOMIC_RELAXED
#define SYS __HIP_MEMORY_SCOPE_SYSTEM
#define AGT __HIP_MEMORY_SCOPE_AGENT

__device__ __forceinline__ bool give_up(unsigned* tmo, unsigned call, u64 t0) {
  if (__hip_atomic_load(tmo, RLX, AGT) == call || __builtin_amdgcn_s_memrealtime() - t0 > DEADLINE) {
    __hip_atomic_store(tmo, call, RLX, AGT);
    return true;
  }
  return false;
}

__device__ __forceinline__ void wait_ge(const unsigned* flag, unsigned target, unsigned* tmo, unsigned call) {
  u64 t0 = __builtin_amdgcn_s_memrealtime();
  while (__hip_atomic_load(flag, RLX, SYS) < target) {
    if (give_up(tmo, call, t0)) break;
    __builtin_amdgcn_s_sleep(2);
  }
}

__global__ __launch_bounds__(NT, 2)
void a2a_gemm_q(int rank, int nranks, const unsigned short* __restrict__ x,
                const unsigned short* __restrict__ w, unsigned char* __restrict__ out, Ptrs P) {
  extern __shared__ __attribute__((aligned(16))) char smem[];
  const int tid = threadIdx.x, lane = tid & 63, wid = tid >> 6;
  const int b = blockIdx.x;
  char* ws = P.ws[rank];
  unsigned* tmo = (unsigned*)(ws + W_TMO);
  volatile unsigned* ctl = (volatile unsigned*)(smem + CTL);

  if (tid == 0) {
    unsigned t = __hip_atomic_fetch_add((unsigned*)(ws + W_CALL), 1u, RLX, AGT);
    ctl[0] = t / NWG + 1;
  }
  if (nranks > 1) {
#pragma unroll
    for (int i = 0; i < 2; ++i) {
      int idx = tid + i * NT;
      unsigned p = (unsigned)(idx >> 7);
      if (p < (unsigned)nranks && (int)p != rank) {
        const u32x4 v = *(const u32x4*)(x + (size_t)p * KSL * KSL + b * 1024 + (idx & 127) * 8);
        *(u32x4*)(P.ws[p] + W_X + (size_t)rank * KSL * KSL * 2 + b * 2048 + (idx & 127) * 16) = v;
      }
    }
    asm volatile("s_waitcnt vmcnt(0)" ::: "memory");
  }
  __syncthreads();
  const unsigned call = ctl[0];
  if (nranks > 1 && tid < nranks && tid != rank) {
    __builtin_amdgcn_fence(__ATOMIC_RELEASE, "");
    asm volatile("s_waitcnt vmcnt(0)" ::: "memory");
    __hip_atomic_fetch_add((unsigned*)(P.ws[tid] + W_XCNT + 128 * rank), 1u, RLX, SYS);
  }

  const int xcd = b & 7, j = b >> 3;
  const int mt = j & 1, nt = xcd * 16 + (j >> 1);
  const int m0 = mt * 256, n0 = nt * 128;
  const int wm = wid >> 1, wn = wid & 1;

  int aoff[4], boff[2];
#pragma unroll
  for (int i = 0; i < 4; ++i) {
    int by = i * 8192 + tid * 16, row = by >> 7, c = ((by >> 4) & 7) ^ (row & 7);
    aoff[i] = (m0 + row) * KSL + c * 8;
  }
#pragma unroll
  for (int i = 0; i < 2; ++i) {
    int by = i * 8192 + tid * 16, k = by >> 8, cp = (by >> 4) & 15;
    int s = (k & 3) | (((k >> 3) & 1) << 2);
    int c = (((cp >> 1) ^ s) << 1) | (cp & 1);
    boff[i] = k * NTOT + n0 + c * 8;
  }
  const int fr = lane & 15, fg = lane >> 4;
  int ardo[2];
#pragma unroll
  for (int kk = 0; kk < 2; ++kk) {
    int row = wm * 64 + fr;
    ardo[kk] = row * 128 + (((kk * 4 + fg) ^ (row & 7)) << 4);
  }
  const int q = (lane >> 2) & 3, p4 = lane & 3;
  int brdo[4];
  {
    int s = q | ((fg & 1) << 2);
#pragma unroll
    for (int ni = 0; ni < 4; ++ni) {
      int c = wn * 8 + ni * 2 + (p4 >> 1);
      brdo[ni] = 32768 + (8 * fg + q) * 256 + ((((c >> 1) ^ s) << 1 | (c & 1)) << 4) + 8 * (p4 & 1);
    }
  }

  f32x4 acc[4][4];
#pragma unroll
  for (int a = 0; a < 4; ++a)
#pragma unroll
    for (int c = 0; c < 4; ++c) acc[a][c] = (f32x4){0.f, 0.f, 0.f, 0.f};

  auto stage = [&](int t, int buf) {
    int sl = (rank + (t >> 3)) & 7;
    const unsigned short* abase = (sl == rank) ? (x + (size_t)rank * KSL * KSL)
                                               : (const unsigned short*)(ws + W_X) + (size_t)sl * KSL * KSL;
    abase += (t & 7) * BK;
    const unsigned short* bbase = w + (size_t)(sl * KSL + (t & 7) * BK) * NTOT;
    char* lb = smem + buf * STG;
#pragma unroll
    for (int i = 0; i < 4; ++i)
      __builtin_amdgcn_global_load_lds((const unsigned*)(abase + aoff[i]), (unsigned*)(lb + i * 8192 + tid * 16), 16, 0, 0);
#pragma unroll
    for (int i = 0; i < 2; ++i)
      __builtin_amdgcn_global_load_lds((const unsigned*)(bbase + boff[i]), (unsigned*)(lb + 32768 + i * 8192 + tid * 16), 16, 0, 0);
  };

  bf16x8 xa[2][2][4];
  s16x4 wl[2][2][4], wh[2][2][4];
  auto rdfrag = [&](int set, int bufi) {
    const unsigned lb = (unsigned)(size_t)smem + bufi * STG;
#pragma unroll
    for (int kk = 0; kk < 2; ++kk) {
      unsigned aa = lb + ardo[kk];
      asm volatile("ds_read_b128 %0, %4\n\tds_read_b128 %1, %4 offset:2048\n\tds_read_b128 %2, %4 offset:4096\n\tds_read_b128 %3, %4 offset:6144"
                   : "=&v"(xa[set][kk][0]), "=&v"(xa[set][kk][1]), "=&v"(xa[set][kk][2]), "=&v"(xa[set][kk][3]) : "v"(aa));
#pragma unroll
      for (int ni = 0; ni < 4; ++ni) {
        unsigned ba = lb + brdo[ni];
        if (kk == 0)
          asm volatile("ds_read_b64_tr_b16 %0, %2\n\tds_read_b64_tr_b16 %1, %2 offset:1024" : "=&v"(wl[set][kk][ni]), "=&v"(wh[set][kk][ni]) : "v"(ba));
        else
          asm volatile("ds_read_b64_tr_b16 %0, %2 offset:8192\n\tds_read_b64_tr_b16 %1, %2 offset:9216" : "=&v"(wl[set][kk][ni]), "=&v"(wh[set][kk][ni]) : "v"(ba));
      }
    }
  };
  auto mma = [&](int set) {
    __builtin_amdgcn_s_setprio(1);
#pragma unroll
    for (int kk = 0; kk < 2; ++kk)
#pragma unroll
      for (int ni = 0; ni < 4; ++ni) {
        bf16x8 wb = (bf16x8){wl[set][kk][ni][0], wl[set][kk][ni][1], wl[set][kk][ni][2], wl[set][kk][ni][3],
                             wh[set][kk][ni][0], wh[set][kk][ni][1], wh[set][kk][ni][2], wh[set][kk][ni][3]};
#pragma unroll
        for (int mi = 0; mi < 4; ++mi)
          acc[ni][mi] = __builtin_amdgcn_mfma_f32_16x16x32_bf16(wb, xa[set][kk][mi], acc[ni][mi], 0, 0, 0);
      }
    __builtin_amdgcn_s_setprio(0);
  };
  auto iter = [&](int t, int set) {
    if (t + 2 < NKT) asm volatile("s_waitcnt vmcnt(6) lgkmcnt(0)" ::: "memory");
    else asm volatile("s_waitcnt vmcnt(0) lgkmcnt(0)" ::: "memory");
    __builtin_amdgcn_sched_barrier(0);
    __builtin_amdgcn_s_barrier();
    if (t + 3 < NKT) {
      stage(t + 3, t % 3);
      if (t == 4 && nranks > 1 && wid == 0) {
        for (int i = 1; i < nranks; ++i)
          wait_ge((const unsigned*)(ws + W_XCNT + 128 * ((rank + i) & 7)), call * NWG, tmo, call);
        __builtin_amdgcn_fence(__ATOMIC_ACQUIRE, "");
        asm volatile("s_waitcnt vmcnt(0)" ::: "memory");
      }
    }
    if (t + 1 < NKT) rdfrag(set ^ 1, (t + 1) % 3);
    __builtin_amdgcn_sched_barrier(0);
    mma(set);
  };
  stage(0, 0);
  stage(1, 1);
  stage(2, 2);
  asm volatile("s_waitcnt vmcnt(12)" ::: "memory");
  __builtin_amdgcn_s_barrier();
  rdfrag(0, 0);
  for (int t = 0; t < NKT; t += 2) {
    iter(t, 0);
    iter(t + 1, 1);
  }

  float mx = 0.f;
#pragma unroll
  for (int a = 0; a < 4; ++a)
#pragma unroll
    for (int c = 0; c < 4; ++c)
#pragma unroll
      for (int e = 0; e < 4; ++e) mx = fmaxf(mx, fabsf(acc[a][c][e]));
#pragma unroll
  for (int o = 32; o >= 1; o >>= 1) mx = fmaxf(mx, __shfl_xor(mx, o));
  if (lane == 0) ctl[8 + wid] = __float_as_uint(mx);
  __syncthreads();
  if (wid == 0) {
    if (lane == 0) {
      unsigned m = 0;
      for (int i = 0; i < 8; ++i) m = max(m, ctl[8 + i]);
      __hip_atomic_fetch_max((u64*)(ws + W_LMAX), ((u64)call << 32) | m, RLX, AGT);
      unsigned tk = __hip_atomic_fetch_add((unsigned*)(ws + W_FIN), 1u, __ATOMIC_ACQ_REL, AGT);
      if (tk == call * NWG - 1) {
        u64 v = __hip_atomic_load((u64*)(ws + W_LMAX), RLX, AGT);
        for (int p = 0; p < nranks; ++p) __hip_atomic_store((u64*)(P.ws[p] + W_SLOT + 128 * rank), v, RLX, SYS);
      }
    }
    unsigned m = 0;
    if (lane < nranks) {
      u64 t0 = __builtin_amdgcn_s_memrealtime();
      u64 v;
      while (((v = __hip_atomic_load((u64*)(ws + W_SLOT + 128 * lane), RLX, SYS)) >> 32) < call) {
        if (give_up(tmo, call, t0)) break;
        __builtin_amdgcn_s_sleep(1);
      }
      m = (unsigned)v;
    }
#pragma unroll
    for (int o = 4; o >= 1; o >>= 1) m = max(m, (unsigned)__shfl_xor((int)m, o));
    if (lane == 0) { ctl[1] = m; ctl[2] = (__hip_atomic_load(tmo, RLX, AGT) == call); }
  }
  __syncthreads();
  const float amax = __uint_as_float(ctl[1]);
  const bool bad = ctl[2] != 0;
  float sc = amax / 448.0f;
  if (!(sc > 0.f)) sc = 1.f;

#pragma unroll
  for (int ni = 0; ni < 4; ++ni)
#pragma unroll
    for (int mi = 0; mi < 4; ++mi) {
      float v0 = fminf(fmaxf(acc[ni][mi][0] / sc, -448.f), 448.f);
      float v1 = fminf(fmaxf(acc[ni][mi][1] / sc, -448.f), 448.f);
      float v2 = fminf(fmaxf(acc[ni][mi][2] / sc, -448.f), 448.f);
      float v3 = fminf(fmaxf(acc[ni][mi][3] / sc, -448.f), 448.f);
      int pk = __builtin_amdgcn_cvt_pk_fp8_f32(v0, v1, 0, false);
      pk = __builtin_amdgcn_cvt_pk_fp8_f32(v2, v3, pk, true);
      if (bad) pk = 0x7f7f7f7f;
      *(int*)(smem + (wm * 64 + mi * 16 + fr) * 144 + wn * 64 + ni * 16 + fg * 4) = pk;
    }
  __syncthreads();
#pragma unroll
  for (int i = 0; i < 4; ++i) {
    int row = i * 64 + (tid >> 3), ch = tid & 7;
    u32x4 v = *(const u32x4*)(smem + row * 144 + ch * 16);
    *(u32x4*)(out + (size_t)(m0 + row) * NTOT + n0 + ch * 16) = v;
  }
}

extern "C" void kernel_launch(int rank, int nranks, ncclComm_t comm, hipStream_t stream,
                              void* const* d_in, const int* in_sizes, int n_in,
                              void* d_out, int out_size, const int* in_dtypes, int out_dtype,
                              void* const* d_out_all, void* const* d_ws_all, size_t ws_bytes) {
  (void)hipFuncSetAttribute((const void*)a2a_gemm_q, hipFuncAttributeMaxDynamicSharedMemorySize, LDS_BYTES);
  Ptrs P;
  for (int p = 0; p < NR; ++p) P.ws[p] = (char*)d_ws_all[p < nranks ? p : rank];
  a2a_gemm_q<<<NWG, NT, LDS_BYTES, stream>>>(rank, nranks, (const unsigned short*)d_in[0],
                                             (const unsigned short*)d_in[1], (unsigned char*)d_out, P);
}
```
